```python
import numpy as np

kernel_name = "gpu_gemm_ar_m4096_k4096_n32768_f32_relu"


ABSMAX_RTOL = 0.001

COMPUTE_DTYPE = "float32"
EPILOGUE = "relu"
IN_QUANT = None
OUT_QUANT = None
IN_DTYPES = {"x": "float32", "w_mat": "float32"}
WORLD_SIZE = 8

def setup_inputs():

    x = np.random.randn(4096, 4096).astype(np.float32)
    w_mat = (np.random.randn(4096, 32768) / np.sqrt(4096)).astype(np.float32)
    return {"x": x, "w_mat": w_mat}

def _epilogue(y):
    return np.maximum(y, 0.0).astype(np.float32)

def reference(x, w_mat):
    y = (x @ w_mat).astype(np.float32)
    return _epilogue(y)
```

```cpp
#include <hip/hip_runtime.h>
#include <rccl/rccl.h>
#include <cstdio>

#define HIP_CHECK(expr) do { hipError_t _e = (expr); if (_e != hipSuccess) { \
    fprintf(stderr, "HIP error %s at %s:%d\n", hipGetErrorString(_e), __FILE__, __LINE__); \
    return; } } while (0)

namespace {

#define LAS __attribute__((address_space(3)))
typedef _Float16 h16_t;
typedef _Float16 h16x8 __attribute__((ext_vector_type(8)));
typedef float f32x4 __attribute__((ext_vector_type(4)));

constexpr int GM = 4096, GN = 32768, GK = 4096, NR = 8, KS = GK / NR;
constexpr int BM = 256, BK = 64, HALF = 128, HTB = HALF * BK * 2, STAGE_BYTES = 8 * HTB;
constexpr int NPM = GM / BM, NPN = GN / BM;
constexpr int NUNITS = NPM + NPN;
constexpr int GRID = 256, NPUSH = 16;
constexpr int CTL_BYTES = 256, LDS_BYTES = STAGE_BYTES + CTL_BYTES;
constexpr int TILES_PER_Q = (NPM * NPN) / 8;

constexpr size_t OFF_CTR = 0;
constexpr size_t OFF_TMO = 128;
constexpr size_t OFF_QCTR = 4096;
constexpr size_t OFF_FLAGS = 16384;
constexpr size_t OFF_DATA = 1u << 20;
constexpr size_t A_BYTES = (size_t)GM * GK * 2, B_BYTES = (size_t)GN * GK * 2, BUF_BYTES = A_BYTES + B_BYTES;
constexpr size_t WS_NEEDED = OFF_DATA + 2 * BUF_BYTES;
constexpr unsigned long long DEADLINE_TICKS = 50000000ull;

struct Args {
    const float* x; const float* w; float* out;
    unsigned char* ws[NR];
    int rank;
};

__device__ __forceinline__ unsigned xcc_id() { return (unsigned)__builtin_amdgcn_s_getreg((3 << 11) | 20) & 0x7u; }

__host__ __device__ __forceinline__ int lds_byte(int r, int c) { const int st = (r >> 4) * 2 + (c >> 5), rr = r & 15, cc = c & 31, ob = rr * 64 + cc * 2; return st * 1024 + (ob ^ (((ob >> 9) & 1) << 5)); }
__host__ __device__ __forceinline__ void stage_rc(int b, int& R, int& C) { const int st = b / 1024, sb = b % 1024, swz = sb ^ (((sb >> 9) & 1) << 5); R = (st >> 1) * 16 + swz / 64; C = (st & 1) * 32 + (swz % 64) / 2; }

__device__ __forceinline__ h16x8 cvt8(const f32x4 a, const f32x4 b) {
    h16x8 r; r[0] = (h16_t)a[0]; r[1] = (h16_t)a[1]; r[2] = (h16_t)a[2]; r[3] = (h16_t)a[3];
    r[4] = (h16_t)b[0]; r[5] = (h16_t)b[1]; r[6] = (h16_t)b[2]; r[7] = (h16_t)b[3]; return r;
}

__device__ __forceinline__ void publish_unit(const Args& a, int unit, unsigned call) {
    asm volatile("s_waitcnt vmcnt(0)" ::: "memory");
    __syncthreads();
    if (threadIdx.x == 0) {
        __builtin_amdgcn_fence(__ATOMIC_RELEASE, "");
        asm volatile("s_waitcnt vmcnt(0)" ::: "memory");
#pragma unroll
        for (int d = 0; d < NR; ++d)
            __hip_atomic_store((unsigned*)(a.ws[d] + OFF_FLAGS) + unit * NR + a.rank, call, __ATOMIC_RELAXED, __HIP_MEMORY_SCOPE_SYSTEM);
    }
}

__device__ __forceinline__ void push_a(const Args& a, int pm, size_t bufoff) {
    const int t = threadIdx.x;
    const size_t dbase = bufoff + ((size_t)pm * BM * GK + (size_t)a.rank * KS) * 2;
#pragma unroll 1
    for (int it = 0; it < 32; it += 4) {
        f32x4 v[4][2];
#pragma unroll
        for (int u = 0; u < 4; ++u) { const int id = (it + u) * 512 + t, row = id >> 6, c = id & 63;
            const f32x4* s = (const f32x4*)(a.x + (size_t)(pm * BM + row) * KS + c * 8); v[u][0] = s[0]; v[u][1] = s[1]; }
#pragma unroll
        for (int u = 0; u < 4; ++u) { const int id = (it + u) * 512 + t, row = id >> 6, c = id & 63;
            const h16x8 h = cvt8(v[u][0], v[u][1]); const size_t off = dbase + ((size_t)row * GK + c * 8) * 2;
#pragma unroll
            for (int j = 0; j < NR; ++j) { const int d = (a.rank + 1 + j + (int)blockIdx.x) & (NR - 1); *(h16x8*)(a.ws[d] + off) = h; } }
    }
}

__device__ __forceinline__ void push_b(const Args& a, int pn, size_t bufoff, LAS unsigned char* lds) {
    const int t = threadIdx.x, wv = t >> 6, l = t & 63;
    const size_t dbase = bufoff + A_BYTES + ((size_t)pn * BM * GK + (size_t)a.rank * KS) * 2;
    const float* src = a.w + (size_t)(wv * 8) * GN + pn * BM + 4 * l;
    f32x4 v[2][8];
#pragma unroll
    for (int p = 0; p < 2; ++p)
#pragma unroll
        for (int j = 0; j < 8; ++j) v[p][j] = *(const f32x4*)(src + (size_t)(p * 64 + j) * GN);
#pragma unroll 1
    for (int sb = 0; sb < 4; ++sb) {
        LAS unsigned char* img = lds + (sb & 1) * 65536;
#pragma unroll
        for (int p = 0; p < 2; ++p)
#pragma unroll
            for (int i = 0; i < 4; ++i) {
                h16x8 h;
#pragma unroll
                for (int j = 0; j < 8; ++j) h[j] = (h16_t)v[p][j][i];
                *(LAS h16x8*)(img + (4 * l + i) * 256 + (((8 * p + wv) ^ (l & 7)) * 16)) = h;
            }
        if (sb < 3) {
#pragma unroll
            for (int p = 0; p < 2; ++p)
#pragma unroll
                for (int j = 0; j < 8; ++j) v[p][j] = *(const f32x4*)(src + (size_t)((sb + 1) * 128 + p * 64 + j) * GN);
        }
        asm volatile("s_waitcnt lgkmcnt(0)" ::: "memory");
        __builtin_amdgcn_s_barrier();
        asm volatile("" ::: "memory");
#pragma unroll
        for (int it = 0; it < 8; ++it) {
            const int id = it * 512 + t, row = id >> 4, c = id & 15;
            const h16x8 h = *(const LAS h16x8*)(img + row * 256 + ((c ^ ((row >> 2) & 7)) * 16));
            const size_t off = dbase + ((size_t)row * GK + sb * 128 + c * 8) * 2;
#pragma unroll
            for (int j = 0; j < NR; ++j) { const int d = (a.rank + 1 + j + (int)blockIdx.x) & (NR - 1); *(h16x8*)(a.ws[d] + off) = h; }
        }
    }
}

struct Ctx {
    unsigned* flags; unsigned* tmo; unsigned* qctr; unsigned call; unsigned xcc;
    LAS int* ctl;
};

__device__ __forceinline__ void draw_tile(const Ctx& c, int& pm, int& pn) {
    pm = -1; pn = -1;
    for (int s = 0; s < 8; ++s) {
        const unsigned q = (c.xcc + s) & 7u;
        const unsigned idx = __hip_atomic_fetch_add(c.qctr + q * 32, 1u, __ATOMIC_RELAXED, __HIP_MEMORY_SCOPE_AGENT);
        if (idx < (unsigned)TILES_PER_Q) { pm = (int)(idx & 15u); pn = (int)(q + 8u * (idx >> 4)); break; }
    }
}

__device__ __forceinline__ void wait_panels(const Ctx& c, int pm, int pn) {
    const int lane = threadIdx.x & 63;
    const unsigned* f = c.flags + ((lane & 8) ? (NPM + pn) : pm) * NR + (lane & 7);
    const unsigned long long t0 = __builtin_amdgcn_s_memrealtime();
    int bad = 0;
    for (;;) {
        const unsigned v = __hip_atomic_load(f, __ATOMIC_RELAXED, __HIP_MEMORY_SCOPE_SYSTEM);
        if (__all(v >= c.call)) break;
        if (__builtin_amdgcn_readfirstlane(__hip_atomic_load(c.tmo, __ATOMIC_RELAXED, __HIP_MEMORY_SCOPE_AGENT)) == c.call
            || __builtin_amdgcn_s_memrealtime() - t0 > DEADLINE_TICKS) {
            if (lane == 0) __hip_atomic_store(c.tmo, c.call, __ATOMIC_RELAXED, __HIP_MEMORY_SCOPE_AGENT);
            bad = 1; break;
        }
        __builtin_amdgcn_s_sleep(2);
    }
    __builtin_amdgcn_fence(__ATOMIC_ACQUIRE, "");
    if (lane == 0) c.ctl[2] = bad;
    asm volatile("s_waitcnt vmcnt(0) lgkmcnt(0)" ::: "memory");
}

__device__ __forceinline__ void gemm_loop(LAS unsigned char* lds, const Ctx& cx, const char* gA, const char* gBt, float* gC) {
    const int tid = threadIdx.x, wid = __builtin_amdgcn_readfirstlane(tid >> 6), lane = tid & 63, wr = wid >> 2, wc = wid & 3, fr = lane & 15, fq = lane >> 4;
    constexpr int nt = GK / BK;
    unsigned voff[2];
#pragma unroll
    for (int i = 0; i < 2; ++i) { int R, C; stage_rc(tid * 16 + i * 8192, R, C); voff[i] = (unsigned)(R * GK + C) * 2u; }
    constexpr size_t kstep = (size_t)(BK * 2);
    constexpr size_t hstep = (size_t)HALF * GK * 2;
    constexpr size_t tstep = 2 * hstep;
    const unsigned ldsw = (unsigned)wid * 1024u;
    const int aoff = lds_byte(wr * 64 + fr, fq * 8), boff = lds_byte(wc * 32 + fr, fq * 8);
#define G_SA(b, h) (((b) * 2 + (h)) * HTB)
#define G_SB(b, h) ((4 + (b) * 2 + (h)) * HTB)
#define G_STAGE(bufoff, gbase) do { _Pragma("unroll") for (int _i = 0; _i < 2; ++_i) \
        __builtin_amdgcn_global_load_lds((const unsigned*)((const char*)(gbase) + voff[_i]), (LAS unsigned*)(lds + (bufoff) + ldsw + _i * 8192), 16, 0, 0); } while (0)
#define G_LDA(dst, b, h) do { _Pragma("unroll") for (int m = 0; m < 4; ++m) _Pragma("unroll") for (int k = 0; k < 2; ++k) dst[m][k] = *(const LAS h16x8*)(lds + G_SA(b, h) + aoff + m * 2048 + k * 1024); } while (0)
#define G_LDB(dst, b, h) do { _Pragma("unroll") for (int n = 0; n < 2; ++n) _Pragma("unroll") for (int k = 0; k < 2; ++k) dst[n][k] = *(const LAS h16x8*)(lds + G_SB(b, h) + boff + n * 2048 + k * 1024); } while (0)
#define G_MMA(ai, bj, At, Bt) do { __builtin_amdgcn_s_setprio(1); _Pragma("unroll") for (int m = 0; m < 4; ++m) _Pragma("unroll") for (int n = 0; n < 2; ++n) _Pragma("unroll") for (int k = 0; k < 2; ++k) \
        acc[ai][bj][m][n] = __builtin_amdgcn_mfma_f32_16x16x32_f16(Bt[n][k], At[m][k], acc[ai][bj][m][n], 0, 0, 0); __builtin_amdgcn_s_setprio(0); } while (0)
#define G_WAIT_V(n) asm volatile("s_waitcnt vmcnt(" #n ")" ::: "memory")
#define G_WAIT_L(n) asm volatile("s_waitcnt lgkmcnt(" #n ")" ::: "memory")
#define G_BAR __builtin_amdgcn_s_barrier()
#define G_SCHED __builtin_amdgcn_sched_barrier(0)

    if (tid == 0) { int pm, pn; draw_tile(cx, pm, pn); cx.ctl[0] = pm; cx.ctl[1] = pn; }
    __syncthreads();
    int cpm = __builtin_amdgcn_readfirstlane(cx.ctl[0]), cpn = __builtin_amdgcn_readfirstlane(cx.ctl[1]);
    if (cpm < 0) return;
    if (wid == 0) wait_panels(cx, cpm, cpn);
    __syncthreads();
    int bad = __builtin_amdgcn_readfirstlane(cx.ctl[2]);
    __syncthreads();

    f32x4 acc[2][2][4][2];
#pragma unroll
    for (int a = 0; a < 2; ++a)
#pragma unroll
        for (int b = 0; b < 2; ++b)
#pragma unroll
            for (int m = 0; m < 4; ++m)
#pragma unroll
                for (int n = 0; n < 2; ++n) acc[a][b][m][n] = (f32x4){0.f, 0.f, 0.f, 0.f};
    h16x8 At[4][2], B0[2][2], B1[2][2];
    const char* cA = gA + (size_t)cpm * tstep; const char* cB = gBt + (size_t)cpn * tstep;
    G_STAGE(G_SB(0, 0), cB); G_STAGE(G_SA(0, 0), cA); G_STAGE(G_SB(0, 1), cB + hstep); G_STAGE(G_SA(0, 1), cA + hstep);
    if (wr == 1) G_BAR;
    G_WAIT_V(4); G_BAR;
    G_STAGE(G_SB(1, 0), cB + kstep); G_STAGE(G_SA(1, 0), cA + kstep); G_STAGE(G_SB(1, 1), cB + hstep + kstep);
    G_WAIT_V(6); G_BAR;
    for (;;) {
        if (tid == 0) { int pm, pn; draw_tile(cx, pm, pn); cx.ctl[0] = pm; cx.ctl[1] = pn; }
        bool has_next = false; int npm = -1, npn = -1;
        const char* nA = cA; const char* nB = cB;
        for (int t = 0; t < nt; t += 2) {
            const bool last = (t == nt - 2);
            if (last) {
                npm = __builtin_amdgcn_readfirstlane(cx.ctl[0]); npn = __builtin_amdgcn_readfirstlane(cx.ctl[1]);
                has_next = npm >= 0;
                if (has_next) {
                    nA = gA + (size_t)npm * tstep; nB = gBt + (size_t)npn * tstep;
                    if (wid == 0) wait_panels(cx, npm, npn);
                    asm volatile("" ::: "memory"); G_BAR; asm volatile("" ::: "memory");
                    bad |= __builtin_amdgcn_readfirstlane(cx.ctl[2]);
                    G_WAIT_L(0);
                }
            }
            const char* a1 = cA + (size_t)(t + 1) * kstep;
            const char* a2 = last ? nA : cA + (size_t)(t + 2) * kstep; const char* b2 = last ? nB : cB + (size_t)(t + 2) * kstep;
            const char* a3 = a2 + kstep; const char* b3 = b2 + kstep;
            G_LDB(B0, 0, 0); G_SCHED; G_LDA(At, 0, 0); G_STAGE(G_SA(1, 1), a1 + hstep);
            G_WAIT_L(8); G_BAR; G_WAIT_L(0); G_MMA(0, 0, At, B0); G_BAR; G_SCHED;
            G_LDB(B1, 0, 1); G_STAGE(G_SB(0, 0), b2);
            G_BAR; G_WAIT_L(0); G_MMA(0, 1, At, B1); G_BAR;
            G_LDA(At, 0, 1); G_STAGE(G_SA(0, 0), a2);
            G_BAR; G_WAIT_L(0); G_MMA(1, 0, At, B0); G_BAR; G_SCHED;
            G_STAGE(G_SB(0, 1), b2 + hstep);
            G_WAIT_V(6); G_BAR; G_MMA(1, 1, At, B1); G_BAR;
            G_LDB(B0, 1, 0); G_SCHED; G_LDA(At, 1, 0); G_STAGE(G_SA(0, 1), a2 + hstep);
            G_WAIT_L(8); G_BAR; G_WAIT_L(0); G_MMA(0, 0, At, B0); G_BAR; G_SCHED;
            G_LDB(B1, 1, 1); G_STAGE(G_SB(1, 0), b3);
            G_BAR; G_WAIT_L(0); G_MMA(0, 1, At, B1); G_BAR;
            G_LDA(At, 1, 1); G_STAGE(G_SA(1, 0), a3);
            G_BAR; G_WAIT_L(0); G_MMA(1, 0, At, B0); G_BAR; G_SCHED;
            G_STAGE(G_SB(1, 1), b3 + hstep);
            G_WAIT_V(6); G_BAR; G_MMA(1, 1, At, B1); G_BAR;
        }
        {
            const float lo = bad ? __builtin_nanf("") : 0.f;
            const int row0 = cpm * BM + wr * 64 + fr, col0 = cpn * BM + wc * 32 + 4 * fq;
#pragma unroll
            for (int ai = 0; ai < 2; ++ai)
#pragma unroll
                for (int m = 0; m < 4; ++m) { float* rowp = gC + (size_t)(row0 + ai * HALF + m * 16) * GN + col0;
#pragma unroll
                    for (int bj = 0; bj < 2; ++bj)
#pragma unroll
                        for (int n = 0; n < 2; ++n) { f32x4 v = acc[ai][bj][m][n];
                            v[0] = bad ? lo : fmaxf(v[0], 0.f); v[1] = bad ? lo : fmaxf(v[1], 0.f); v[2] = bad ? lo : fmaxf(v[2], 0.f); v[3] = bad ? lo : fmaxf(v[3], 0.f);
                            *(f32x4*)(rowp + bj * HALF + n * 16) = v; } }
        }
        if (!has_next) break;
#pragma unroll
        for (int a = 0; a < 2; ++a)
#pragma unroll
            for (int b = 0; b < 2; ++b)
#pragma unroll
                for (int m = 0; m < 4; ++m)
#pragma unroll
                    for (int n = 0; n < 2; ++n) acc[a][b][m][n] = (f32x4){0.f, 0.f, 0.f, 0.f};
        cpm = npm; cpn = npn; cA = nA; cB = nB;
    }
    G_WAIT_V(0);
    if (wr == 0) G_BAR;
    G_BAR;
#undef G_SA
#undef G_SB
#undef G_STAGE
#undef G_LDA
#undef G_LDB
#undef G_MMA
#undef G_WAIT_V
#undef G_WAIT_L
#undef G_BAR
#undef G_SCHED
}

__global__ __launch_bounds__(512, 2) void fused_gemm_kernel(Args a) {
    extern __shared__ __attribute__((aligned(16))) unsigned char shm[];
    LAS unsigned char* lds = (LAS unsigned char*)shm;
    LAS int* ctl = (LAS int*)(lds + STAGE_BYTES);
    unsigned char* myws = a.ws[a.rank];
    if (threadIdx.x == 0) {
        const unsigned ticket = __hip_atomic_fetch_add((unsigned*)(myws + OFF_CTR), 1u, __ATOMIC_RELAXED, __HIP_MEMORY_SCOPE_AGENT);
        ctl[3] = (int)(ticket / gridDim.x + 1u);
    }
    __syncthreads();
    const unsigned call = (unsigned)__builtin_amdgcn_readfirstlane(ctl[3]);
    const size_t bufoff = OFF_DATA + (size_t)(call & 1u) * BUF_BYTES;

    Ctx cx;
    cx.flags = (unsigned*)(myws + OFF_FLAGS); cx.tmo = (unsigned*)(myws + OFF_TMO);
    cx.qctr = (unsigned*)(myws + OFF_QCTR + (size_t)(call & 1u) * 1024); cx.call = call; cx.xcc = xcc_id(); cx.ctl = ctl;

    if (blockIdx.x < (unsigned)NPUSH) {
        if (blockIdx.x == 0 && threadIdx.x < 8)
            __hip_atomic_store((unsigned*)(myws + OFF_QCTR + (size_t)((call + 1u) & 1u) * 1024) + threadIdx.x * 32, 0u, __ATOMIC_RELAXED, __HIP_MEMORY_SCOPE_AGENT);
        for (int u = (int)blockIdx.x; u < NUNITS; u += NPUSH) {
            if (u < 2 * NPM) { if (u & 1) { push_b(a, u >> 1, bufoff, lds); publish_unit(a, NPM + (u >> 1), call); }
                               else { push_a(a, u >> 1, bufoff); publish_unit(a, u >> 1, call); } }
            else { push_b(a, u - NPM, bufoff, lds); publish_unit(a, u, call); }
        }
        __syncthreads();
    }
    gemm_loop(lds, cx, (const char*)(myws + bufoff), (const char*)(myws + bufoff + A_BYTES), a.out);
}

}

extern "C" void kernel_launch(int rank, int nranks, ncclComm_t comm,
                              hipStream_t stream,
                              void* const* d_in, const int* in_sizes, int n_in,
                              void* d_out, int out_size,
                              const int* in_dtypes, int out_dtype,
                              void* const* d_out_all,
                              void* const* d_ws_all, size_t ws_bytes) {
    if (nranks != NR || n_in < 2 || ws_bytes < WS_NEEDED || (size_t)in_sizes[0] != (size_t)GM * KS || (size_t)in_sizes[1] != (size_t)KS * GN) {
        fprintf(stderr, "kernel_launch: unexpected configuration (nranks %d, ws %zu)\n", nranks, ws_bytes);
        return;
    }
    static bool attr_set = false;
    if (!attr_set) { HIP_CHECK(hipFuncSetAttribute((const void*)fused_gemm_kernel, hipFuncAttributeMaxDynamicSharedMemorySize, LDS_BYTES)); attr_set = true; }
    Args a;
    a.x = (const float*)d_in[0]; a.w = (const float*)d_in[1]; a.out = (float*)d_out; a.rank = rank;
    for (int p = 0; p < NR; ++p) a.ws[p] = (unsigned char*)d_ws_all[p];
    hipLaunchKernelGGL(fused_gemm_kernel, dim3(GRID), dim3(512), LDS_BYTES, stream, a);
    HIP_CHECK(hipGetLastError());
}
```

```cpp
#include <hip/hip_runtime.h>
#include <rccl/rccl.h>
#include <cstdio>

#define HIP_CHECK(expr) do { hipError_t _e = (expr); if (_e != hipSuccess) { \
    fprintf(stderr, "HIP error %s at %s:%d\n", hipGetErrorString(_e), __FILE__, __LINE__); \
    return; } } while (0)

namespace {

#define LAS __attribute__((address_space(3)))
typedef _Float16 h16_t;
typedef _Float16 h16x8 __attribute__((ext_vector_type(8)));
typedef float f32x4 __attribute__((ext_vector_type(4)));

constexpr int GM = 4096, GN = 32768, GK = 4096, NR = 8, KS = GK / NR;
constexpr int BM = 256, BK = 64, HALF = 128, HTB = HALF * BK * 2, STAGE_BYTES = 8 * HTB;
constexpr int NPM = GM / BM, NPN = GN / BM;
constexpr int NUNITS = NPM + NPN;
constexpr int GRID = 256, NPUSH = 16;
constexpr int CTL_BYTES = 256, LDS_BYTES = STAGE_BYTES + CTL_BYTES;
constexpr int TILES_PER_Q = (NPM * NPN) / 8;

constexpr size_t OFF_CTR = 0;
constexpr size_t OFF_TMO = 128;
constexpr size_t OFF_QCTR = 4096;
constexpr size_t OFF_FLAGS = 16384;
constexpr size_t OFF_DATA = 1u << 20;
constexpr size_t A_BYTES = (size_t)GM * GK * 2, B_BYTES = (size_t)GN * GK * 2, BUF_BYTES = A_BYTES + B_BYTES;
constexpr size_t WS_NEEDED = OFF_DATA + 2 * BUF_BYTES;
constexpr unsigned long long DEADLINE_TICKS = 50000000ull;

struct Args {
    const float* x; const float* w; float* out;
    unsigned char* ws[NR];
    int rank;
};
#define STAMP(i) do {} while (0)

__device__ __forceinline__ unsigned xcc_id() { return (unsigned)__builtin_amdgcn_s_getreg((3 << 11) | 20) & 0x7u; }

__host__ __device__ __forceinline__ int lds_byte(int r, int c) { const int st = (r >> 4) * 2 + (c >> 5), rr = r & 15, cc = c & 31, ob = rr * 64 + cc * 2; return st * 1024 + (ob ^ (((ob >> 9) & 1) << 5)); }
__host__ __device__ __forceinline__ void stage_rc(int b, int& R, int& C) { const int st = b / 1024, sb = b % 1024, swz = sb ^ (((sb >> 9) & 1) << 5); R = (st >> 1) * 16 + swz / 64; C = (st & 1) * 32 + (swz % 64) / 2; }

__device__ __forceinline__ h16x8 cvt8(const f32x4 a, const f32x4 b) {
    h16x8 r; r[0] = (h16_t)a[0]; r[1] = (h16_t)a[1]; r[2] = (h16_t)a[2]; r[3] = (h16_t)a[3];
    r[4] = (h16_t)b[0]; r[5] = (h16_t)b[1]; r[6] = (h16_t)b[2]; r[7] = (h16_t)b[3]; return r;
}

#define PUSH_STORE(p, v) asm volatile("global_store_dwordx4 %0, %1, off sc0 sc1\n\ts_nop 1" :: "v"(p), "v"(v) : "memory")
__device__ __forceinline__ void publish_piece(const Args& a, int piece, unsigned call) {
    asm volatile("s_waitcnt vmcnt(0)" ::: "memory");
    __syncthreads();
    if (threadIdx.x == 0) {
#pragma unroll
        for (int d = 0; d < NR; ++d)
            __hip_atomic_store((unsigned*)(a.ws[d] + OFF_FLAGS) + piece * NR + a.rank, call, __ATOMIC_RELAXED, __HIP_MEMORY_SCOPE_SYSTEM);
    }
}

__device__ __forceinline__ void push_a(const Args& a, int q, size_t bufoff) {
    const int t = threadIdx.x;
    const size_t dbase = bufoff + ((size_t)q * 64 * GK + (size_t)a.rank * KS) * 2;
    f32x4 v[8][2];
#pragma unroll
    for (int u = 0; u < 8; ++u) { const int id = u * 512 + t, row = id >> 6, c = id & 63;
        const f32x4* s = (const f32x4*)(a.x + (size_t)(q * 64 + row) * KS + c * 8); v[u][0] = s[0]; v[u][1] = s[1]; }
#pragma unroll
    for (int u = 0; u < 8; ++u) { const int id = u * 512 + t, row = id >> 6, c = id & 63;
        const h16x8 h = cvt8(v[u][0], v[u][1]); const size_t off = dbase + ((size_t)row * GK + c * 8) * 2;
#pragma unroll
        for (int j = 0; j < NR; ++j) { const int d = (a.rank + 1 + j + (int)blockIdx.x) & (NR - 1); PUSH_STORE((h16x8*)(a.ws[d] + off), h); } }
}

__device__ __forceinline__ void push_b(const Args& a, int q, size_t bufoff, LAS unsigned char* lds) {
    const int t = threadIdx.x, nq = t & 15, kg = t >> 4;
    const size_t dbase = bufoff + A_BYTES + ((size_t)q * 64 * GK + (size_t)a.rank * KS) * 2;
    const float* src = a.w + (size_t)(kg * 16) * GN + q * 64 + 4 * nq;
    f32x4 v[16];
#pragma unroll
    for (int j = 0; j < 16; ++j) v[j] = *(const f32x4*)(src + (size_t)j * GN);
#pragma unroll
    for (int p = 0; p < 2; ++p)
#pragma unroll
        for (int i = 0; i < 4; ++i) {
            h16x8 h;
#pragma unroll
            for (int j = 0; j < 8; ++j) h[j] = (h16_t)v[p * 8 + j][i];
            *(LAS h16x8*)(lds + (4 * nq + i) * 1024 + (((2 * kg + p) ^ (nq & 7)) * 16)) = h;
        }
    asm volatile("s_waitcnt lgkmcnt(0)" ::: "memory");
    __builtin_amdgcn_s_barrier();
    asm volatile("" ::: "memory");
#pragma unroll
    for (int it = 0; it < 8; ++it) {
        const int id = it * 512 + t, row = id >> 6, c = id & 63;
        const h16x8 h = *(const LAS h16x8*)(lds + row * 1024 + ((c ^ ((row >> 2) & 7)) * 16));
        const size_t off = dbase + ((size_t)row * GK + c * 8) * 2;
#pragma unroll
        for (int j = 0; j < NR; ++j) { const int d = (a.rank + 1 + j + (int)blockIdx.x) & (NR - 1); PUSH_STORE((h16x8*)(a.ws[d] + off), h); }
    }
}

struct Ctx {
    unsigned* flags; unsigned* tmo; unsigned* qctr; unsigned call; unsigned xcc;
    LAS int* ctl;
};

__device__ __forceinline__ void draw_tile(const Ctx& c, int& pm, int& pn) {
    pm = -1; pn = -1;
    for (int s = 0; s < 8; ++s) {
        const unsigned q = (c.xcc + s) & 7u;
        const unsigned idx = __hip_atomic_fetch_add(c.qctr + q * 32, 1u, __ATOMIC_RELAXED, __HIP_MEMORY_SCOPE_AGENT);
        if (idx < (unsigned)TILES_PER_Q) { pm = (int)(idx & 15u); pn = (int)(q + 8u * (idx >> 4)); break; }
    }
}

__device__ __forceinline__ void wait_panels(const Ctx& c, int pm, int pn) {
    const int lane = threadIdx.x & 63;
    const unsigned* f = c.flags + (((lane & 32) ? (NPM + pn) : pm) * 4 + ((lane >> 3) & 3)) * NR + (lane & 7);
    const unsigned long long t0 = __builtin_amdgcn_s_memrealtime();
    int bad = 0;
    for (;;) {
        const unsigned v = __hip_atomic_load(f, __ATOMIC_RELAXED, __HIP_MEMORY_SCOPE_SYSTEM);
        if (__all(v >= c.call)) break;
        if (__builtin_amdgcn_readfirstlane(__hip_atomic_load(c.tmo, __ATOMIC_RELAXED, __HIP_MEMORY_SCOPE_AGENT)) == c.call
            || __builtin_amdgcn_s_memrealtime() - t0 > DEADLINE_TICKS) {
            if (lane == 0) __hip_atomic_store(c.tmo, c.call, __ATOMIC_RELAXED, __HIP_MEMORY_SCOPE_AGENT);
            bad = 1; break;
        }
        __builtin_amdgcn_s_sleep(2);
    }
    __builtin_amdgcn_fence(__ATOMIC_ACQUIRE, "");
    if (lane == 0) c.ctl[2] = bad;
    asm volatile("s_waitcnt vmcnt(0) lgkmcnt(0)" ::: "memory");
}

__device__ __forceinline__ void gemm_loop(LAS unsigned char* lds, const Ctx& cx, const char* gA, const char* gBt, float* gC) {
    const int tid = threadIdx.x, wid = __builtin_amdgcn_readfirstlane(tid >> 6), lane = tid & 63, wr = wid >> 2, wc = wid & 3, fr = lane & 15, fq = lane >> 4;
    constexpr int nt = GK / BK;
    unsigned voff[2];
#pragma unroll
    for (int i = 0; i < 2; ++i) { int R, C; stage_rc(tid * 16 + i * 8192, R, C); voff[i] = (unsigned)(R * GK + C) * 2u; }
    constexpr size_t kstep = (size_t)(BK * 2);
    constexpr size_t hstep = (size_t)HALF * GK * 2;
    constexpr size_t tstep = 2 * hstep;
    const unsigned ldsw = (unsigned)wid * 1024u;
    const int aoff = lds_byte(wr * 64 + fr, fq * 8), boff = lds_byte(wc * 32 + fr, fq * 8);
#define G_SA(b, h) (((b) * 2 + (h)) * HTB)
#define G_SB(b, h) ((4 + (b) * 2 + (h)) * HTB)
#define G_STAGE(bufoff, gbase) do { _Pragma("unroll") for (int _i = 0; _i < 2; ++_i) \
        __builtin_amdgcn_global_load_lds((const unsigned*)((const char*)(gbase) + voff[_i]), (LAS unsigned*)(lds + (bufoff) + ldsw + _i * 8192), 16, 0, 0); } while (0)
#define G_LDA(dst, b, h) do { _Pragma("unroll") for (int m = 0; m < 4; ++m) _Pragma("unroll") for (int k = 0; k < 2; ++k) dst[m][k] = *(const LAS h16x8*)(lds + G_SA(b, h) + aoff + m * 2048 + k * 1024); } while (0)
#define G_LDB(dst, b, h) do { _Pragma("unroll") for (int n = 0; n < 2; ++n) _Pragma("unroll") for (int k = 0; k < 2; ++k) dst[n][k] = *(const LAS h16x8*)(lds + G_SB(b, h) + boff + n * 2048 + k * 1024); } while (0)
#define G_MMA(ai, bj, At, Bt) do { __builtin_amdgcn_s_setprio(1); _Pragma("unroll") for (int m = 0; m < 4; ++m) _Pragma("unroll") for (int n = 0; n < 2; ++n) _Pragma("unroll") for (int k = 0; k < 2; ++k) \
        acc[ai][bj][m][n] = __builtin_amdgcn_mfma_f32_16x16x32_f16(Bt[n][k], At[m][k], acc[ai][bj][m][n], 0, 0, 0); __builtin_amdgcn_s_setprio(0); } while (0)
#define G_WAIT_V(n) asm volatile("s_waitcnt vmcnt(" #n ")" ::: "memory")
#define G_WAIT_L(n) asm volatile("s_waitcnt lgkmcnt(" #n ")" ::: "memory")
#define G_BAR __builtin_amdgcn_s_barrier()
#define G_SCHED __builtin_amdgcn_sched_barrier(0)

    if (tid == 0) { int pm, pn; draw_tile(cx, pm, pn); cx.ctl[0] = pm; cx.ctl[1] = pn; }
    __syncthreads();
    int cpm = __builtin_amdgcn_readfirstlane(cx.ctl[0]), cpn = __builtin_amdgcn_readfirstlane(cx.ctl[1]);
    if (cpm < 0) return;
    if (wid == 0) wait_panels(cx, cpm, cpn);
    __syncthreads();
    int bad = __builtin_amdgcn_readfirstlane(cx.ctl[2]);
    __syncthreads();
    STAMP(ns);

    f32x4 acc[2][2][4][2];
#pragma unroll
    for (int a = 0; a < 2; ++a)
#pragma unroll
        for (int b = 0; b < 2; ++b)
#pragma unroll
            for (int m = 0; m < 4; ++m)
#pragma unroll
                for (int n = 0; n < 2; ++n) acc[a][b][m][n] = (f32x4){0.f, 0.f, 0.f, 0.f};
    h16x8 At[4][2], B0[2][2], B1[2][2];
    const char* cA = gA + (size_t)cpm * tstep; const char* cB = gBt + (size_t)cpn * tstep;
    G_STAGE(G_SB(0, 0), cB); G_STAGE(G_SA(0, 0), cA); G_STAGE(G_SB(0, 1), cB + hstep); G_STAGE(G_SA(0, 1), cA + hstep);
    if (wr == 1) G_BAR;
    G_WAIT_V(4); G_BAR;
    G_STAGE(G_SB(1, 0), cB + kstep); G_STAGE(G_SA(1, 0), cA + kstep); G_STAGE(G_SB(1, 1), cB + hstep + kstep);
    G_WAIT_V(6); G_BAR;
    for (;;) {
        if (tid == 0) { int pm, pn; draw_tile(cx, pm, pn); cx.ctl[0] = pm; cx.ctl[1] = pn; }
        bool has_next = false; int npm = -1, npn = -1;
        const char* nA = cA; const char* nB = cB;
        for (int t = 0; t < nt; t += 2) {
            const bool last = (t == nt - 2);
            if (last) {
                npm = __builtin_amdgcn_readfirstlane(cx.ctl[0]); npn = __builtin_amdgcn_readfirstlane(cx.ctl[1]);
                has_next = npm >= 0;
                if (has_next) {
                    nA = gA + (size_t)npm * tstep; nB = gBt + (size_t)npn * tstep;
                    STAMP(ns);
                    if (wid == 0) wait_panels(cx, npm, npn);
                    STAMP(ns);
                    asm volatile("" ::: "memory"); G_BAR; asm volatile("" ::: "memory");
                    bad |= __builtin_amdgcn_readfirstlane(cx.ctl[2]);
                    G_WAIT_L(0);
                }
            }
            const char* a1 = cA + (size_t)(t + 1) * kstep;
            const char* a2 = last ? nA : cA + (size_t)(t + 2) * kstep; const char* b2 = last ? nB : cB + (size_t)(t + 2) * kstep;
            const char* a3 = a2 + kstep; const char* b3 = b2 + kstep;
            G_LDB(B0, 0, 0); G_SCHED; G_LDA(At, 0, 0); G_STAGE(G_SA(1, 1), a1 + hstep);
            G_WAIT_L(8); G_BAR; G_WAIT_L(0); G_MMA(0, 0, At, B0); G_BAR; G_SCHED;
            G_LDB(B1, 0, 1); G_STAGE(G_SB(0, 0), b2);
            G_BAR; G_WAIT_L(0); G_MMA(0, 1, At, B1); G_BAR;
            G_LDA(At, 0, 1); G_STAGE(G_SA(0, 0), a2);
            G_BAR; G_WAIT_L(0); G_MMA(1, 0, At, B0); G_BAR; G_SCHED;
            G_STAGE(G_SB(0, 1), b2 + hstep);
            G_WAIT_V(6); G_BAR; G_MMA(1, 1, At, B1); G_BAR;
            G_LDB(B0, 1, 0); G_SCHED; G_LDA(At, 1, 0); G_STAGE(G_SA(0, 1), a2 + hstep);
            G_WAIT_L(8); G_BAR; G_WAIT_L(0); G_MMA(0, 0, At, B0); G_BAR; G_SCHED;
            G_LDB(B1, 1, 1); G_STAGE(G_SB(1, 0), b3);
            G_BAR; G_WAIT_L(0); G_MMA(0, 1, At, B1); G_BAR;
            G_LDA(At, 1, 1); G_STAGE(G_SA(1, 0), a3);
            G_BAR; G_WAIT_L(0); G_MMA(1, 0, At, B0); G_BAR; G_SCHED;
            G_STAGE(G_SB(1, 1), b3 + hstep);
            G_WAIT_V(6); G_BAR; G_MMA(1, 1, At, B1); G_BAR;
        }
        {
            const float lo = bad ? __builtin_nanf("") : 0.f;
            const int row0 = cpm * BM + wr * 64 + fr, col0 = cpn * BM + wc * 32 + 4 * fq;
#pragma unroll
            for (int ai = 0; ai < 2; ++ai)
#pragma unroll
                for (int m = 0; m < 4; ++m) { float* rowp = gC + (size_t)(row0 + ai * HALF + m * 16) * GN + col0;
#pragma unroll
                    for (int bj = 0; bj < 2; ++bj)
#pragma unroll
                        for (int n = 0; n < 2; ++n) { f32x4 v = acc[ai][bj][m][n];
                            v[0] = bad ? lo : fmaxf(v[0], 0.f); v[1] = bad ? lo : fmaxf(v[1], 0.f); v[2] = bad ? lo : fmaxf(v[2], 0.f); v[3] = bad ? lo : fmaxf(v[3], 0.f);
                            *(f32x4*)(rowp + bj * HALF + n * 16) = v; } }
        }
        STAMP(ns);
        if (!has_next) break;
#pragma unroll
        for (int a = 0; a < 2; ++a)
#pragma unroll
            for (int b = 0; b < 2; ++b)
#pragma unroll
                for (int m = 0; m < 4; ++m)
#pragma unroll
                    for (int n = 0; n < 2; ++n) acc[a][b][m][n] = (f32x4){0.f, 0.f, 0.f, 0.f};
        cpm = npm; cpn = npn; cA = nA; cB = nB;
    }
    G_WAIT_V(0);
    if (wr == 0) G_BAR;
    G_BAR;
#undef G_SA
#undef G_SB
#undef G_STAGE
#undef G_LDA
#undef G_LDB
#undef G_MMA
#undef G_WAIT_V
#undef G_WAIT_L
#undef G_BAR
#undef G_SCHED
}

__global__ __launch_bounds__(512, 2) void fused_gemm_kernel(Args a) {
    extern __shared__ __attribute__((aligned(16))) unsigned char shm[];
    LAS unsigned char* lds = (LAS unsigned char*)shm;
    LAS int* ctl = (LAS int*)(lds + STAGE_BYTES);
    unsigned char* myws = a.ws[a.rank];
    if (threadIdx.x == 0) {
        const unsigned ticket = __hip_atomic_fetch_add((unsigned*)(myws + OFF_CTR), 1u, __ATOMIC_RELAXED, __HIP_MEMORY_SCOPE_AGENT);
        ctl[3] = (int)(ticket / gridDim.x + 1u);
    }
    __syncthreads();
    const unsigned call = (unsigned)__builtin_amdgcn_readfirstlane(ctl[3]);
    const size_t bufoff = OFF_DATA + (size_t)(call & 1u) * BUF_BYTES;

    Ctx cx;
    cx.flags = (unsigned*)(myws + OFF_FLAGS); cx.tmo = (unsigned*)(myws + OFF_TMO);
    cx.qctr = (unsigned*)(myws + OFF_QCTR + (size_t)(call & 1u) * 1024); cx.call = call; cx.xcc = xcc_id(); cx.ctl = ctl;

    const int pidx = (int)blockIdx.x;
    if (pidx < NPUSH) {
        if (blockIdx.x == 0 && threadIdx.x < 8)
            __hip_atomic_store((unsigned*)(myws + OFF_QCTR + (size_t)((call + 1u) & 1u) * 1024) + threadIdx.x * 32, 0u, __ATOMIC_RELAXED, __HIP_MEMORY_SCOPE_AGENT);
        for (int u = pidx; u < NUNITS * 4; u += NPUSH) {
            const int unit = u >> 2, qq = u & 3;
            if (unit < 2 * NPM) { if (unit & 1) { push_b(a, (unit >> 1) * 4 + qq, bufoff, lds); publish_piece(a, (NPM + (unit >> 1)) * 4 + qq, call); }
                                  else { push_a(a, (unit >> 1) * 4 + qq, bufoff); publish_piece(a, (unit >> 1) * 4 + qq, call); } }
            else { push_b(a, (unit - NPM) * 4 + qq, bufoff, lds); publish_piece(a, unit * 4 + qq, call); }
            STAMP(ns);
        }
        __syncthreads();
    }
    gemm_loop(lds, cx, (const char*)(myws + bufoff), (const char*)(myws + bufoff + A_BYTES), a.out);
}

}

extern "C" void kernel_launch(int rank, int nranks, ncclComm_t comm,
                              hipStream_t stream,
                              void* const* d_in, const int* in_sizes, int n_in,
                              void* d_out, int out_size,
                              const int* in_dtypes, int out_dtype,
                              void* const* d_out_all,
                              void* const* d_ws_all, size_t ws_bytes) {
    if (nranks != NR || n_in < 2 || ws_bytes < WS_NEEDED || (size_t)in_sizes[0] != (size_t)GM * KS || (size_t)in_sizes[1] != (size_t)KS * GN) {
        fprintf(stderr, "kernel_launch: unexpected configuration (nranks %d, ws %zu)\n", nranks, ws_bytes);
        return;
    }
    static bool attr_set = false;
    if (!attr_set) { HIP_CHECK(hipFuncSetAttribute((const void*)fused_gemm_kernel, hipFuncAttributeMaxDynamicSharedMemorySize, LDS_BYTES)); attr_set = true; }
    Args a;
    a.x = (const float*)d_in[0]; a.w = (const float*)d_in[1]; a.out = (float*)d_out; a.rank = rank;
    for (int p = 0; p < NR; ++p) a.ws[p] = (unsigned char*)d_ws_all[p];
    hipLaunchKernelGGL(fused_gemm_kernel, dim3(GRID), dim3(512), LDS_BYTES, stream, a);
    HIP_CHECK(hipGetLastError());
}
```

```cpp
#include <hip/hip_runtime.h>
#include <rccl/rccl.h>
#include <cstdio>

#define HIP_CHECK(expr) do { hipError_t _e = (expr); if (_e != hipSuccess) { \
    fprintf(stderr, "HIP error %s at %s:%d\n", hipGetErrorString(_e), __FILE__, __LINE__); \
    return; } } while (0)

namespace {

#define LAS __attribute__((address_space(3)))
typedef _Float16 h16_t;
typedef _Float16 h16x8 __attribute__((ext_vector_type(8)));
typedef float f32x4 __attribute__((ext_vector_type(4)));

constexpr int GM = 4096, GN = 32768, GK = 4096, NR = 8, KS = GK / NR;
constexpr int BM = 256, BK = 64, HALF = 128, HTB = HALF * BK * 2, STAGE_BYTES = 8 * HTB;
constexpr int NPM = GM / BM, NPN = GN / BM;
constexpr int NUNITS = NPM + NPN;
constexpr int GRID = 256, NPUSH = 16;
constexpr int CTL_BYTES = 256, LDS_BYTES = STAGE_BYTES + CTL_BYTES;
constexpr int TILES_PER_Q = (NPM * NPN) / 8;

constexpr size_t OFF_CTR = 0;
constexpr size_t OFF_TMO = 128;
constexpr size_t OFF_QCTR = 4096;
constexpr size_t OFF_FLAGS = 16384;
constexpr size_t OFF_DATA = 1u << 20;
constexpr size_t A_BYTES = (size_t)GM * GK * 2, B_BYTES = (size_t)GN * GK * 2, BUF_BYTES = A_BYTES + B_BYTES;
constexpr size_t WS_NEEDED = OFF_DATA + 2 * BUF_BYTES;
constexpr unsigned long long DEADLINE_TICKS = 50000000ull;

struct Args {
    const float* x; const float* w; float* out;
    unsigned char* ws[NR];
    int rank;
};
#define STAMP(i) do {} while (0)

__device__ __forceinline__ unsigned xcc_id() { return (unsigned)__builtin_amdgcn_s_getreg((3 << 11) | 20) & 0x7u; }

__host__ __device__ __forceinline__ int lds_byte(int r, int c) { const int st = (r >> 4) * 2 + (c >> 5), rr = r & 15, cc = c & 31, ob = rr * 64 + cc * 2; return st * 1024 + (ob ^ (((ob >> 9) & 1) << 5)); }
__host__ __device__ __forceinline__ void stage_rc(int b, int& R, int& C) { const int st = b / 1024, sb = b % 1024, swz = sb ^ (((sb >> 9) & 1) << 5); R = (st >> 1) * 16 + swz / 64; C = (st & 1) * 32 + (swz % 64) / 2; }

__device__ __forceinline__ h16x8 cvt8(const f32x4 a, const f32x4 b) {
    h16x8 r; r[0] = (h16_t)a[0]; r[1] = (h16_t)a[1]; r[2] = (h16_t)a[2]; r[3] = (h16_t)a[3];
    r[4] = (h16_t)b[0]; r[5] = (h16_t)b[1]; r[6] = (h16_t)b[2]; r[7] = (h16_t)b[3]; return r;
}

#define PUSH_STORE(p, v) asm volatile("global_store_dwordx4 %0, %1, off sc0 sc1\n\ts_nop 1" :: "v"(p), "v"(v) : "memory")
__device__ __forceinline__ void publish_piece(const Args& a, int piece, unsigned call) {
    asm volatile("s_waitcnt vmcnt(0)" ::: "memory");
    __syncthreads();
    if (threadIdx.x == 0) {
#pragma unroll
        for (int d = 0; d < NR; ++d)
            __hip_atomic_store((unsigned*)(a.ws[d] + OFF_FLAGS) + piece * NR + a.rank, call, __ATOMIC_RELAXED, __HIP_MEMORY_SCOPE_SYSTEM);
    }
}

__device__ __forceinline__ void push_a(const Args& a, int q, size_t bufoff) {
    const int t = threadIdx.x;
    const size_t dbase = bufoff + ((size_t)q * 64 * GK + (size_t)a.rank * KS) * 2;
    f32x4 v[8][2];
#pragma unroll
    for (int u = 0; u < 8; ++u) { const int id = u * 512 + t, row = id >> 6, c = id & 63;
        const f32x4* s = (const f32x4*)(a.x + (size_t)(q * 64 + row) * KS + c * 8); v[u][0] = s[0]; v[u][1] = s[1]; }
#pragma unroll
    for (int u = 0; u < 8; ++u) { const int id = u * 512 + t, row = id >> 6, c = id & 63;
        const h16x8 h = cvt8(v[u][0], v[u][1]); const size_t off = dbase + ((size_t)row * GK + c * 8) * 2;
#pragma unroll
        for (int j = 0; j < NR; ++j) { const int d = (a.rank + 1 + j + (int)blockIdx.x) & (NR - 1); PUSH_STORE((h16x8*)(a.ws[d] + off), h); } }
}

__device__ __forceinline__ void push_b(const Args& a, int q, size_t bufoff, LAS unsigned char* lds) {
    const int t = threadIdx.x, nq = t & 15, kg = t >> 4;
    const size_t dbase = bufoff + A_BYTES + ((size_t)q * 64 * GK + (size_t)a.rank * KS) * 2;
    const float* src = a.w + (size_t)(kg * 16) * GN + q * 64 + 4 * nq;
    f32x4 v[16];
#pragma unroll
    for (int j = 0; j < 16; ++j) v[j] = *(const f32x4*)(src + (size_t)j * GN);
#pragma unroll
    for (int p = 0; p < 2; ++p)
#pragma unroll
        for (int i = 0; i < 4; ++i) {
            h16x8 h;
#pragma unroll
            for (int j = 0; j < 8; ++j) h[j] = (h16_t)v[p * 8 + j][i];
            *(LAS h16x8*)(lds + (4 * nq + i) * 1024 + (((2 * kg + p) ^ (nq & 7)) * 16)) = h;
        }
    asm volatile("s_waitcnt lgkmcnt(0)" ::: "memory");
    __builtin_amdgcn_s_barrier();
    asm volatile("" ::: "memory");
#pragma unroll
    for (int it = 0; it < 8; ++it) {
        const int id = it * 512 + t, row = id >> 6, c = id & 63;
        const h16x8 h = *(const LAS h16x8*)(lds + row * 1024 + ((c ^ ((row >> 2) & 7)) * 16));
        const size_t off = dbase + ((size_t)row * GK + c * 8) * 2;
#pragma unroll
        for (int j = 0; j < NR; ++j) { const int d = (a.rank + 1 + j + (int)blockIdx.x) & (NR - 1); PUSH_STORE((h16x8*)(a.ws[d] + off), h); }
    }
}

struct Ctx {
    unsigned* flags; unsigned* tmo; unsigned* qctr; unsigned call; unsigned xcc;
    LAS int* ctl;
};

__device__ __forceinline__ void draw_tile(const Ctx& c, int& pm, int& pn) {
    pm = -1; pn = -1;
    for (int s = 0; s < 8; ++s) {
        const unsigned q = (c.xcc + s) & 7u;
        const unsigned idx = __hip_atomic_fetch_add(c.qctr + q * 32, 1u, __ATOMIC_RELAXED, __HIP_MEMORY_SCOPE_AGENT);
        if (idx < (unsigned)TILES_PER_Q) { pm = (int)(8u * (idx & 1u) + q); pn = (int)(idx >> 1); break; }
    }
}

__device__ __forceinline__ void wait_panels(const Ctx& c, int pm, int pn) {
    const int lane = threadIdx.x & 63;
    const unsigned* f = c.flags + (((lane & 32) ? (NPM + pn) : pm) * 4 + ((lane >> 3) & 3)) * NR + (lane & 7);
    const unsigned long long t0 = __builtin_amdgcn_s_memrealtime();
    int bad = 0;
    for (;;) {
        const unsigned v = __hip_atomic_load(f, __ATOMIC_RELAXED, __HIP_MEMORY_SCOPE_SYSTEM);
        if (__all(v >= c.call)) break;
        if (__builtin_amdgcn_readfirstlane(__hip_atomic_load(c.tmo, __ATOMIC_RELAXED, __HIP_MEMORY_SCOPE_AGENT)) == c.call
            || __builtin_amdgcn_s_memrealtime() - t0 > DEADLINE_TICKS) {
            if (lane == 0) __hip_atomic_store(c.tmo, c.call, __ATOMIC_RELAXED, __HIP_MEMORY_SCOPE_AGENT);
            bad = 1; break;
        }
        __builtin_amdgcn_s_sleep(2);
    }
    __builtin_amdgcn_fence(__ATOMIC_ACQUIRE, "");
    if (lane == 0) c.ctl[2] = bad;
    asm volatile("s_waitcnt vmcnt(0) lgkmcnt(0)" ::: "memory");
}

__device__ __forceinline__ void gemm_loop(LAS unsigned char* lds, const Ctx& cx, const char* gA, const char* gBt, float* gC) {
    const int tid = threadIdx.x, wid = __builtin_amdgcn_readfirstlane(tid >> 6), lane = tid & 63, wr = wid >> 2, wc = wid & 3, fr = lane & 15, fq = lane >> 4;
    constexpr int nt = GK / BK;
    unsigned voff[2];
#pragma unroll
    for (int i = 0; i < 2; ++i) { int R, C; stage_rc(tid * 16 + i * 8192, R, C); voff[i] = (unsigned)(R * GK + C) * 2u; }
    constexpr size_t kstep = (size_t)(BK * 2);
    constexpr size_t hstep = (size_t)HALF * GK * 2;
    constexpr size_t tstep = 2 * hstep;
    const unsigned ldsw = (unsigned)wid * 1024u;
    const int aoff = lds_byte(wr * 64 + fr, fq * 8), boff = lds_byte(wc * 32 + fr, fq * 8);
#define G_SA(b, h) (((b) * 2 + (h)) * HTB)
#define G_SB(b, h) ((4 + (b) * 2 + (h)) * HTB)
#define G_STAGE(bufoff, gbase) do { _Pragma("unroll") for (int _i = 0; _i < 2; ++_i) \
        __builtin_amdgcn_global_load_lds((const unsigned*)((const char*)(gbase) + voff[_i]), (LAS unsigned*)(lds + (bufoff) + ldsw + _i * 8192), 16, 0, 0); } while (0)
#define G_LDA(dst, b, h) do { _Pragma("unroll") for (int m = 0; m < 4; ++m) _Pragma("unroll") for (int k = 0; k < 2; ++k) dst[m][k] = *(const LAS h16x8*)(lds + G_SA(b, h) + aoff + m * 2048 + k * 1024); } while (0)
#define G_LDB(dst, b, h) do { _Pragma("unroll") for (int n = 0; n < 2; ++n) _Pragma("unroll") for (int k = 0; k < 2; ++k) dst[n][k] = *(const LAS h16x8*)(lds + G_SB(b, h) + boff + n * 2048 + k * 1024); } while (0)
#define G_MMA(ai, bj, At, Bt) do { __builtin_amdgcn_s_setprio(1); _Pragma("unroll") for (int m = 0; m < 4; ++m) _Pragma("unroll") for (int n = 0; n < 2; ++n) _Pragma("unroll") for (int k = 0; k < 2; ++k) \
        acc[ai][bj][m][n] = __builtin_amdgcn_mfma_f32_16x16x32_f16(At[m][k], Bt[n][k], acc[ai][bj][m][n], 0, 0, 0); __builtin_amdgcn_s_setprio(0); } while (0)
#define G_WAIT_V(n) asm volatile("s_waitcnt vmcnt(" #n ")" ::: "memory")
#define G_WAIT_L(n) asm volatile("s_waitcnt lgkmcnt(" #n ")" ::: "memory")
#define G_BAR __builtin_amdgcn_s_barrier()
#define G_SCHED __builtin_amdgcn_sched_barrier(0)

    if (tid == 0) { int pm, pn; draw_tile(cx, pm, pn); cx.ctl[0] = pm; cx.ctl[1] = pn; }
    __syncthreads();
    int cpm = __builtin_amdgcn_readfirstlane(cx.ctl[0]), cpn = __builtin_amdgcn_readfirstlane(cx.ctl[1]);
    if (cpm < 0) return;
    if (wid == 0) wait_panels(cx, cpm, cpn);
    __syncthreads();
    int bad = __builtin_amdgcn_readfirstlane(cx.ctl[2]);
    __syncthreads();
    STAMP(ns);

    f32x4 acc[2][2][4][2];
#pragma unroll
    for (int a = 0; a < 2; ++a)
#pragma unroll
        for (int b = 0; b < 2; ++b)
#pragma unroll
            for (int m = 0; m < 4; ++m)
#pragma unroll
                for (int n = 0; n < 2; ++n) acc[a][b][m][n] = (f32x4){0.f, 0.f, 0.f, 0.f};
    h16x8 At[4][2], B0[2][2], B1[2][2];
    const char* cA = gA + (size_t)cpm * tstep; const char* cB = gBt + (size_t)cpn * tstep;
    G_STAGE(G_SB(0, 0), cB); G_STAGE(G_SA(0, 0), cA); G_STAGE(G_SB(0, 1), cB + hstep); G_STAGE(G_SA(0, 1), cA + hstep);
    if (wr == 1) G_BAR;
    G_WAIT_V(4); G_BAR;
    G_STAGE(G_SB(1, 0), cB + kstep); G_STAGE(G_SA(1, 0), cA + kstep); G_STAGE(G_SB(1, 1), cB + hstep + kstep);
    G_WAIT_V(6); G_BAR;
    for (;;) {
        if (tid == 0) { int pm, pn; draw_tile(cx, pm, pn); cx.ctl[0] = pm; cx.ctl[1] = pn; }
        bool has_next = false; int npm = -1, npn = -1;
        const char* nA = cA; const char* nB = cB;
        for (int t = 0; t < nt; t += 2) {
            const bool last = (t == nt - 2);
            if (last) {
                npm = __builtin_amdgcn_readfirstlane(cx.ctl[0]); npn = __builtin_amdgcn_readfirstlane(cx.ctl[1]);
                has_next = npm >= 0;
                if (has_next) {
                    nA = gA + (size_t)npm * tstep; nB = gBt + (size_t)npn * tstep;
                    STAMP(ns);
                    if (wid == 0) wait_panels(cx, npm, npn);
                    STAMP(ns);
                    asm volatile("" ::: "memory"); G_BAR; asm volatile("" ::: "memory");
                    bad |= __builtin_amdgcn_readfirstlane(cx.ctl[2]);
                    G_WAIT_L(0);
                }
            }
            const char* a1 = cA + (size_t)(t + 1) * kstep;
            const char* a2 = last ? nA : cA + (size_t)(t + 2) * kstep; const char* b2 = last ? nB : cB + (size_t)(t + 2) * kstep;
            const char* a3 = a2 + kstep; const char* b3 = b2 + kstep;
            G_LDB(B0, 0, 0); G_SCHED; G_LDA(At, 0, 0); G_STAGE(G_SA(1, 1), a1 + hstep);
            G_WAIT_L(8); G_BAR; G_WAIT_L(0); G_MMA(0, 0, At, B0); G_BAR; G_SCHED;
            G_LDB(B1, 0, 1); G_STAGE(G_SB(0, 0), b2);
            G_BAR; G_WAIT_L(0); G_MMA(0, 1, At, B1); G_BAR;
            G_LDA(At, 0, 1); G_STAGE(G_SA(0, 0), a2);
            G_BAR; G_WAIT_L(0); G_MMA(1, 0, At, B0); G_BAR; G_SCHED;
            G_STAGE(G_SB(0, 1), b2 + hstep);
            G_WAIT_V(6); G_BAR; G_MMA(1, 1, At, B1); G_BAR;
            G_LDB(B0, 1, 0); G_SCHED; G_LDA(At, 1, 0); G_STAGE(G_SA(0, 1), a2 + hstep);
            G_WAIT_L(8); G_BAR; G_WAIT_L(0); G_MMA(0, 0, At, B0); G_BAR; G_SCHED;
            G_LDB(B1, 1, 1); G_STAGE(G_SB(1, 0), b3);
            G_BAR; G_WAIT_L(0); G_MMA(0, 1, At, B1); G_BAR;
            G_LDA(At, 1, 1); G_STAGE(G_SA(1, 0), a3);
            G_BAR; G_WAIT_L(0); G_MMA(1, 0, At, B0); G_BAR; G_SCHED;
            G_STAGE(G_SB(1, 1), b3 + hstep);
            G_WAIT_V(6); G_BAR; G_MMA(1, 1, At, B1); G_BAR;
        }
        {
            const float lo = bad ? __builtin_nanf("") : 0.f;
            const int row0 = cpm * BM + wr * 64 + 4 * fq, col0 = cpn * BM + wc * 32 + fr;
#pragma unroll
            for (int ai = 0; ai < 2; ++ai)
#pragma unroll
                for (int m = 0; m < 4; ++m)
#pragma unroll
                    for (int q = 0; q < 4; ++q) { float* rowp = gC + (size_t)(row0 + ai * HALF + m * 16 + q) * GN + col0;
#pragma unroll
                        for (int bj = 0; bj < 2; ++bj)
#pragma unroll
                            for (int n = 0; n < 2; ++n) rowp[bj * HALF + n * 16] = bad ? lo : fmaxf(acc[ai][bj][m][n][q], 0.f); }
        }
        STAMP(ns);
        if (!has_next) break;
#pragma unroll
        for (int a = 0; a < 2; ++a)
#pragma unroll
            for (int b = 0; b < 2; ++b)
#pragma unroll
                for (int m = 0; m < 4; ++m)
#pragma unroll
                    for (int n = 0; n < 2; ++n) acc[a][b][m][n] = (f32x4){0.f, 0.f, 0.f, 0.f};
        cpm = npm; cpn = npn; cA = nA; cB = nB;
    }
    G_WAIT_V(0);
    if (wr == 0) G_BAR;
    G_BAR;
#undef G_SA
#undef G_SB
#undef G_STAGE
#undef G_LDA
#undef G_LDB
#undef G_MMA
#undef G_WAIT_V
#undef G_WAIT_L
#undef G_BAR
#undef G_SCHED
}

__global__ __launch_bounds__(512, 2) void fused_gemm_kernel(Args a) {
    extern __shared__ __attribute__((aligned(16))) unsigned char shm[];
    LAS unsigned char* lds = (LAS unsigned char*)shm;
    LAS int* ctl = (LAS int*)(lds + STAGE_BYTES);
    unsigned char* myws = a.ws[a.rank];
    if (threadIdx.x == 0) {
        const unsigned ticket = __hip_atomic_fetch_add((unsigned*)(myws + OFF_CTR), 1u, __ATOMIC_RELAXED, __HIP_MEMORY_SCOPE_AGENT);
        ctl[3] = (int)(ticket / gridDim.x + 1u);
    }
    __syncthreads();
    const unsigned call = (unsigned)__builtin_amdgcn_readfirstlane(ctl[3]);
    const size_t bufoff = OFF_DATA + (size_t)(call & 1u) * BUF_BYTES;

    Ctx cx;
    cx.flags = (unsigned*)(myws + OFF_FLAGS); cx.tmo = (unsigned*)(myws + OFF_TMO);
    cx.qctr = (unsigned*)(myws + OFF_QCTR + (size_t)(call & 1u) * 1024); cx.call = call; cx.xcc = xcc_id(); cx.ctl = ctl;

    const int pidx = (int)blockIdx.x;
    if (pidx < NPUSH) {
        if (blockIdx.x == 0 && threadIdx.x < 8)
            __hip_atomic_store((unsigned*)(myws + OFF_QCTR + (size_t)((call + 1u) & 1u) * 1024) + threadIdx.x * 32, 0u, __ATOMIC_RELAXED, __HIP_MEMORY_SCOPE_AGENT);
        for (int u = pidx; u < NUNITS * 4; u += NPUSH) {
            const int unit = u >> 2, qq = u & 3;
            if (unit < 2 * NPM) { if (unit & 1) { push_b(a, (unit >> 1) * 4 + qq, bufoff, lds); publish_piece(a, (NPM + (unit >> 1)) * 4 + qq, call); }
                                  else { push_a(a, (unit >> 1) * 4 + qq, bufoff); publish_piece(a, (unit >> 1) * 4 + qq, call); } }
            else { push_b(a, (unit - NPM) * 4 + qq, bufoff, lds); publish_piece(a, unit * 4 + qq, call); }
            STAMP(ns);
        }
        __syncthreads();
    }
    gemm_loop(lds, cx, (const char*)(myws + bufoff), (const char*)(myws + bufoff + A_BYTES), a.out);
}

}

extern "C" void kernel_launch(int rank, int nranks, ncclComm_t comm,
                              hipStream_t stream,
                              void* const* d_in, const int* in_sizes, int n_in,
                              void* d_out, int out_size,
                              const int* in_dtypes, int out_dtype,
                              void* const* d_out_all,
                              void* const* d_ws_all, size_t ws_bytes) {
    if (nranks != NR || n_in < 2 || ws_bytes < WS_NEEDED || (size_t)in_sizes[0] != (size_t)GM * KS || (size_t)in_sizes[1] != (size_t)KS * GN) {
        fprintf(stderr, "kernel_launch: unexpected configuration (nranks %d, ws %zu)\n", nranks, ws_bytes);
        return;
    }
    static bool attr_set = false;
    if (!attr_set) { HIP_CHECK(hipFuncSetAttribute((const void*)fused_gemm_kernel, hipFuncAttributeMaxDynamicSharedMemorySize, LDS_BYTES)); attr_set = true; }
    Args a;
    a.x = (const float*)d_in[0]; a.w = (const float*)d_in[1]; a.out = (float*)d_out; a.rank = rank;
    for (int p = 0; p < NR; ++p) a.ws[p] = (unsigned char*)d_ws_all[p];
    hipLaunchKernelGGL(fused_gemm_kernel, dim3(GRID), dim3(512), LDS_BYTES, stream, a);
    HIP_CHECK(hipGetLastError());
}
```
